```python
import numpy as np

kernel_name = "gpu_gemm_rs_m4096_k4096_n4096_bf16_silu"


ABSMAX_RTOL = 0.008

COMPUTE_DTYPE = "bfloat16"
EPILOGUE = "silu"
IN_QUANT = None
OUT_QUANT = None
IN_DTYPES = {"x": "bfloat16", "w_mat": "bfloat16"}
WORLD_SIZE = 8

def _to_bf16(x):
    u = np.ascontiguousarray(x, dtype=np.float32).view(np.uint32)
    return ((u + 0x7FFF + ((u >> 16) & 1)) & 0xFFFF0000).view(np.float32)

def setup_inputs():

    x = _to_bf16(np.random.randn(4096, 4096).astype(np.float32))
    w_mat = _to_bf16((np.random.randn(4096, 4096) / np.sqrt(4096)).astype(np.float32))
    return {"x": x, "w_mat": w_mat}

def _epilogue(y):
    return (y / (1.0 + np.exp(-np.clip(y, -60.0, 60.0)))).astype(np.float32)

def reference(x, w_mat):
    y = (x @ w_mat).astype(np.float32)
    return _epilogue(y)
```

```cpp
#include <hip/hip_runtime.h>
#include <rccl/rccl.h>
#include <cstdio>

typedef unsigned u32;
typedef unsigned long long u64;
typedef u32 u32x2 __attribute__((ext_vector_type(2)));
typedef u32 u32x4 __attribute__((ext_vector_type(4)));
typedef unsigned short u16x2 __attribute__((ext_vector_type(2)));
typedef short s16x4 __attribute__((ext_vector_type(4)));
typedef short s16x8 __attribute__((ext_vector_type(8)));
typedef float f32x2 __attribute__((ext_vector_type(2)));
typedef float f32x4 __attribute__((ext_vector_type(4)));
typedef __bf16 bf16x2 __attribute__((ext_vector_type(2)));
typedef __amdgpu_buffer_rsrc_t rsrc_t;
#define LAS __attribute__((address_space(3)))
#define GAS __attribute__((address_space(1)))

namespace {

constexpr int NR = 8;
constexpr int NWG = 256;
constexpr int NC = 32;
constexpr int NW = NC * 2;
constexpr int IT1 = 512 / NW, IT2 = 256 / NW, ITO = 256 / NC;
constexpr int SL2 = 6 * IT1 * 2, SLO = SL2 + 6 * IT2 * 2, NSL = SLO + ITO * 2;

constexpr u32 OFF_CNT = 0;
constexpr u32 OFF_TMO = 16u << 10;
constexpr u32 OFF_RF = 32u << 10;
constexpr u32 OFF_H1 = 64u << 10;
constexpr u32 OFF_H2 = 128u << 10;
constexpr u32 OFF_DBG = 256u << 10;
constexpr u32 PBYTES = 8u << 20;
constexpr u32 OFF_PI = 1u << 20;
constexpr u32 OFF_PC = OFF_PI + 2 * PBYTES;
constexpr u32 OFF_UI = OFF_PC + 2 * PBYTES;
constexpr u32 UIBYTES = 12u << 20;
constexpr u32 WS_NEED = OFF_UI + UIBYTES;
constexpr u32 IN_BYTES = 4u << 20;
constexpr u32 OOB = 0x7ffffff0u;
constexpr u64 DEADLINE = 100000000ull;
constexpr u32 H_VALID = 0x80000000u, H_POISON = 0x40000000u;

constexpr int LDS_W = 65536, LDS_CTL = 131072, LDS_BYTES = LDS_CTL + 1024;

__device__ constexpr unsigned ORD1[128] = {0,4,64,68,8,12,72,76,80,84,88,92,1,5,65,69,9,13,73,77,81,85,89,93,2,6,66,70,10,14,74,78,82,86,90,94,16,20,3,7,67,71,11,15,24,28,75,79,83,87,32,36,91,95,17,21,96,25,29,100,40,44,33,37,18,22,97,104,26,30,101,108,41,45,34,38,48,52,19,23,98,105,27,31,102,112,109,42,46,35,39,49,53,116,99,106,56,60,103,113,110,43,47,50,54,117,120,107,57,61,124,114,111,51,55,118,121,58,62,125,115,119,122,59,63,126,123,127};
__device__ constexpr unsigned ORD2[128] = {0,1,16,17,2,18,3,19,4,20,5,21,6,22,7,23,32,33,34,35,36,37,38,39,48,49,50,51,52,53,54,55,64,65,66,67,68,69,70,71,8,24,40,56,72,9,25,41,57,73,80,81,82,83,84,85,86,87,88,89,10,26,42,58,74,90,11,27,43,59,75,91,96,97,98,99,100,101,102,103,104,105,106,107,12,28,44,60,76,92,108,13,29,45,61,77,93,109,112,113,114,115,116,117,118,119,120,121,122,123,124,125,14,30,46,62,78,94,110,126,15,31,47,63,79,95,111,127};
__device__ constexpr unsigned TILES[224] = {17694720,18743312,19791904,20840496,17760257,18808849,19857441,20906033,21889026,22937618,23986210,25034802,21954563,23003155,24051747,25100339,26083332,27131924,28180516,29229108,26148869,27197461,28246053,29294645,30277638,31326230,32374822,33423414,30343175,31391767,32440359,33488951,64,80,96,112,65,81,97,113,66,82,98,114,67,83,99,115,68,84,100,116,69,85,101,117,70,86,102,118,71,87,103,119,128,144,40,56,129,145,41,57,130,146,104,120,131,147,105,121,132,148,160,176,133,149,161,177,134,150,162,178,135,151,163,179,8,24,164,180,9,25,165,181,72,88,166,182,73,89,167,183,136,152,168,184,137,153,169,185,10,26,42,58,11,27,43,59,74,90,106,122,75,91,107,123,138,154,170,186,139,155,171,187,192,208,44,60,193,209,45,61,194,210,108,124,195,211,109,125,196,212,172,188,197,213,173,189,198,214,224,240,199,215,225,241,200,216,226,242,201,217,227,243,202,218,228,244,203,219,229,245,12,28,230,246,13,29,231,247,76,92,232,248,77,93,233,249,140,156,234,250,141,157,235,251,204,220,236,252,205,221,237,253};

struct Params {
    char* ws[NR];
    const char* x;
    const char* w;
    char* out;
    int rank;
    int pad;
};

#define DEVI __device__ __forceinline__
#define WAIT_VM(n) asm volatile("s_waitcnt vmcnt(" #n ")" ::: "memory")

DEVI rsrc_t mk_rsrc(const void* p, u32 bytes) { return __builtin_amdgcn_make_buffer_rsrc((void*)p, 0, bytes, 0x00020000); }
DEVI u32x4 mk_desc(const void* p, u32 bytes) {
    const u64 a = (u64)p;
    return (u32x4){(u32)a, (u32)(a >> 32) & 0xffffu, bytes, 0x00020000u};
}
DEVI u32 uni(u32 v) { return __builtin_amdgcn_readfirstlane(v); }
DEVI u32 bfi(u32 m, u32 a, u32 b) { return (a & m) | (b & ~m); }
DEVI u32 cvt_pk(float lo, float hi) {
    const bf16x2 r = __builtin_convertvector((f32x2){lo, hi}, bf16x2);
    return __builtin_bit_cast(u32, r);
}
DEVI u32 umax(u32 a, u32 b) { return a > b ? a : b; }
DEVI u32 wave_max(u32 v) {
    v = umax(v, (u32)__builtin_amdgcn_update_dpp(0, (int)v, 0x111, 0xf, 0xf, false));
    v = umax(v, (u32)__builtin_amdgcn_update_dpp(0, (int)v, 0x112, 0xf, 0xf, false));
    v = umax(v, (u32)__builtin_amdgcn_update_dpp(0, (int)v, 0x114, 0xf, 0xf, false));
    v = umax(v, (u32)__builtin_amdgcn_update_dpp(0, (int)v, 0x118, 0xf, 0xf, false));
    v = umax(v, (u32)__builtin_amdgcn_update_dpp(0, (int)v, 0x142, 0xa, 0xf, false));
    v = umax(v, (u32)__builtin_amdgcn_update_dpp(0, (int)v, 0x143, 0xc, 0xf, false));
    return (u32)__builtin_amdgcn_readlane((int)v, 63);
}

#define STAMP(idx) do { } while (0)

DEVI u32 pack_store(const u32 (&d)[16], rsrc_t r, u32 off, int lane) {
    u16x2 mx = __builtin_bit_cast(u16x2, d[0] & 0x7fff7fffu), mn = mx;
#pragma unroll
    for (int i = 1; i < 16; ++i) {
        const u16x2 t = __builtin_bit_cast(u16x2, d[i] & 0x7fff7fffu);
        mx = __builtin_elementwise_max(mx, t);
        mn = __builtin_elementwise_min(mn, t);
    }
    const u32 emax = (u32)(mx.x > mx.y ? mx.x : mx.y) >> 8, emin = (u32)(mn.x < mn.y ? mn.x : mn.y) >> 8;
    const u32 wmax = wave_max(emax);
    const u32 base = wmax > 7u ? wmax - 7u : 0u;
    const u64 bal = __builtin_amdgcn_ballot_w64(emin < base);
    u64 quad;
    asm("s_quadmask_b64 %0, %1" : "=s"(quad) : "s"(bal) : "scc");
    const u32 mask = (u32)quad;
    const bool exc = (mask >> (lane >> 2)) & 1u;
    const u32 brep = exc ? 0u : base * 0x01010101u;
    u32 L[8], H[8];
#pragma unroll
    for (int i = 0; i < 8; ++i) {
        L[i] = __builtin_amdgcn_perm(d[2 * i + 1], d[2 * i], 0x06040200u);
        H[i] = __builtin_amdgcn_perm(d[2 * i + 1], d[2 * i], 0x07050301u) - brep;
    }
    u32x4 A, Bn;
#pragma unroll
    for (int i = 0; i < 4; ++i) {
        const u32 ro = __builtin_amdgcn_alignbit(H[2 * i + 1], H[2 * i + 1], 28);
        A[i] = bfi(0x87878787u, H[2 * i], ro);
        Bn[i] = bfi(0x87878787u, ro, H[2 * i]);
    }
    __builtin_amdgcn_raw_buffer_store_b128((u32x4){L[0], L[1], L[2], L[3]}, r, off, 0, 17);
    __builtin_amdgcn_raw_buffer_store_b128((u32x4){L[4], L[5], L[6], L[7]}, r, off + 1024u, 0, 17);
    __builtin_amdgcn_raw_buffer_store_b128(A, r, off + 2048u, 0, 17);
    __builtin_amdgcn_raw_buffer_store_b128(Bn, r, exc ? off + 3072u : OOB, 0, 17);
    return H_VALID | (base << 16) | mask;
}

struct Blk { u32x4 l0, l1, a, b; };
DEVI Blk load_block(rsrc_t r, u32 off, u32 hdr, int lane) {
    Blk k;
    k.l0 = __builtin_amdgcn_raw_buffer_load_b128(r, off, 0, 17);
    k.l1 = __builtin_amdgcn_raw_buffer_load_b128(r, off + 1024u, 0, 17);
    k.a = __builtin_amdgcn_raw_buffer_load_b128(r, off + 2048u, 0, 17);
    k.b = __builtin_amdgcn_raw_buffer_load_b128(r, ((hdr >> (lane >> 2)) & 1u) ? off + 3072u : OOB, 0, 17);
    return k;
}
DEVI void unpack(const Blk& k, u32 hdr, int lane, u32 (&d)[16]) {
    const u32 brep = ((hdr >> (lane >> 2)) & 1u) ? 0u : ((hdr >> 16) & 0x7fu) * 0x01010101u;
#pragma unroll
    for (int i = 0; i < 4; ++i) {
        const u32 he = bfi(0x87878787u, k.a[i], k.b[i]) + brep;
        const u32 t = bfi(0x87878787u, k.b[i], k.a[i]);
        const u32 ho = __builtin_amdgcn_alignbit(t, t, 4) + brep;
        const u32 le = i < 2 ? k.l0[2 * i] : k.l1[2 * i - 4], lo = i < 2 ? k.l0[2 * i + 1] : k.l1[2 * i - 3];
        d[4 * i + 0] = __builtin_amdgcn_perm(he, le, 0x05010400u);
        d[4 * i + 1] = __builtin_amdgcn_perm(he, le, 0x07030602u);
        d[4 * i + 2] = __builtin_amdgcn_perm(ho, lo, 0x05010400u);
        d[4 * i + 3] = __builtin_amdgcn_perm(ho, lo, 0x07030602u);
    }
}

DEVI float silu(float y) {
    const float c = __builtin_amdgcn_fmed3f(y, -60.0f, 60.0f);
    const float e = __builtin_amdgcn_exp2f(c * -1.44269504088896340736f);
    return y * __builtin_amdgcn_rcpf(1.0f + e);
}

DEVI u32 x_piece(u32 half, u32 panel, u32 p, int lane) {
    const u32 r = p * 8u + ((u32)lane >> 3);
    const u32 grow = (4u * (panel & 1u) + (r >> 5)) * 512u + 256u * half + 32u * (panel >> 1) + (r & 31u);
    return grow * 1024u + ((((u32)lane & 7u) ^ ((r >> 1) & 7u)) << 4);
}
DEVI u32 w_piece(u32 half, u32 panel, u32 p, int lane) {
    const u32 k = p * 4u + ((u32)lane >> 4);
    return k * 8192u + (2048u * half + 128u * panel) * 2u + ((((u32)lane & 15u) ^ (((k & 3u) << 2) | ((k >> 2) & 3u))) << 4);
}

DEVI u32x2 wait_slots(LAS u32* slots) {
    u64 v;
    for (;;) {
        v = __hip_atomic_load((LAS u64*)slots, __ATOMIC_ACQUIRE, __HIP_MEMORY_SCOPE_WORKGROUP);
        const u32 a = uni((u32)v), b = uni((u32)(v >> 32));
        if (a & b & H_VALID) return (u32x2){a, b};
        __builtin_amdgcn_s_sleep(1);
    }
}

__global__ void __launch_bounds__(512, 2) fused(Params p) {
    __shared__ __attribute__((aligned(16))) char smem_[LDS_BYTES];
    LAS char* smem = (LAS char*)smem_;
    const int lane = threadIdx.x & 63;
    const u32 wv = uni(threadIdx.x >> 6);
    const u32 B = blockIdx.x;
    const u32 rank = (u32)p.rank & 7u;
    char* me = p.ws[rank];
    GAS u32* tmo = (GAS u32*)(me + OFF_TMO);
    LAS u32* ctl = (LAS u32*)(smem + LDS_CTL);
    STAMP(0);

    GAS u32* cnt = (GAS u32*)(me + OFF_CNT + B * 64u);
    const u32 cnt_v = __hip_atomic_load(cnt, __ATOMIC_RELAXED, __HIP_MEMORY_SCOPE_AGENT);
    if (threadIdx.x < 256) ctl[threadIdx.x] = 0u;

    u32 call, par;
    {
        const u32 dc = ORD1[B >> 1], half = B & 1u, h = wv >> 2, b = wv & 3u;
        const u32 op = dc >> 6, panel = (dc >> 2) & 15u, kk = h * 4u + (dc & 3u);
        const rsrc_t rs = mk_rsrc(op ? p.w : p.x, IN_BYTES);
        const u32 so = op ? kk * 524288u : kk * 128u;
        u32 d[16];
#pragma unroll
        for (int u = 0; u < 4; ++u) {
            const u32 o = op ? w_piece(half, panel, 4u * b + u, lane) : x_piece(half, panel, 4u * b + u, lane);
            const u32x4 v = __builtin_amdgcn_raw_buffer_load_b128(rs, o, so, 0);
            d[4 * u] = v[0]; d[4 * u + 1] = v[1]; d[4 * u + 2] = v[2]; d[4 * u + 3] = v[3];
        }
        call = uni(cnt_v) + 1u;
        par = call & 1u;
        const u32 ci = ((op * 2u + half) * 16u + panel) * 8u + kk;
        const u32 hdr = pack_store(d, mk_rsrc(me + OFF_PI + par * PBYTES, PBYTES), (ci * 4u + b) * 4096u + lane * 16u, lane);
        const u32 d0 = (h << 2) | (op ? half : half << 1), d1 = d0 | (op ? 2u : 1u);
        char* q0 = p.ws[d0 & 7u];
        char* q1 = p.ws[d1 & 7u];
        asm volatile("" : "+s"(q0), "+s"(q1));
        WAIT_VM(0);
        const u32 dst = lane ? d1 : d0;
        if (lane < 2 && (dst & 3u) != (rank & 3u)) {
            GAS u64* g = (GAS u64*)((lane ? q1 : q0) + OFF_H1 + ((par * 8u + rank) * 128u + dc) * 32u + b * 8u);
            __hip_atomic_store(g, ((u64)hdr << 32) | call, __ATOMIC_RELAXED, __HIP_MEMORY_SCOPE_SYSTEM);
        }
    }
    STAMP(1);
    asm volatile("s_waitcnt lgkmcnt(0)" ::: "memory");
    __builtin_amdgcn_s_barrier();
    asm volatile("" ::: "memory");
    if (threadIdx.x == 0) __hip_atomic_store(cnt, call, __ATOMIC_RELAXED, __HIP_MEMORY_SCOPE_AGENT);

    if (B < (u32)NC) {
        if (wv == 7u) {
            constexpr int NP = (NSL + 63) / 64;
            GAS u64* g[NP];
            bool todo[NP];
#pragma unroll
            for (int c = 0; c < NP; ++c) {
                const u32 s = c * 64u + lane, e = s & 1u;
                u32 off;
                if (s < (u32)SL2) {
                    const u32 w = (s >> 1) / IT1, it = (s >> 1) % IT1, n = B * 2u + w / 3u + NW * it;
                    const u32 src = (e << 2) | ((rank ^ (w % 3u + 1u)) & 3u);
                    off = OFF_H1 + ((par * 8u + src) * 128u + ORD1[(n >> 2) & 127u]) * 32u + (n & 3u) * 8u;
                } else {
                    u32 n, q;
                    if (s < (u32)SLO) {
                        const u32 w = ((s - SL2) >> 1) / IT2, it = ((s - SL2) >> 1) % IT2;
                        n = B * 2u + w / 3u + NW * it; q = (rank ^ (w % 3u + 1u)) & 3u;
                    } else {
                        n = B + NC * ((s - SLO) >> 1); q = rank & 3u;
                    }
                    off = OFF_H2 + ((par * 8u + ((e << 2) | q)) * 256u + ORD2[(n >> 1) & 127u] * 2u + (n & 1u)) * 8u;
                }
                g[c] = (GAS u64*)(me + off);
                todo[c] = s < (u32)NSL;
            }
            const u64 t0 = __builtin_amdgcn_s_memrealtime();
            for (;;) {
                u64 v[NP];
#pragma unroll
                for (int c = 0; c < NP; ++c) v[c] = todo[c] ? __hip_atomic_load(g[c], __ATOMIC_RELAXED, __HIP_MEMORY_SCOPE_SYSTEM) : 0ull;
                const u32 tv = __hip_atomic_load(tmo, __ATOMIC_RELAXED, __HIP_MEMORY_SCOPE_AGENT);
                const bool dead = uni(tv) == call || __builtin_amdgcn_s_memrealtime() - t0 > DEADLINE;
                if (dead && lane == 0) __hip_atomic_store(tmo, call, __ATOMIC_RELAXED, __HIP_MEMORY_SCOPE_AGENT);
                bool left = false;
#pragma unroll
                for (int c = 0; c < NP; ++c) {
                    if (todo[c] && (dead || (u32)v[c] == call)) {
                        __hip_atomic_store(ctl + c * 64 + lane, dead ? (H_VALID | H_POISON) : (u32)(v[c] >> 32), __ATOMIC_RELEASE, __HIP_MEMORY_SCOPE_WORKGROUP);
                        todo[c] = false;
                    }
                    left |= todo[c];
                }
                if (__builtin_amdgcn_ballot_w64(left) == 0ull) break;
                __builtin_amdgcn_s_sleep(2);
            }
        } else {
            const bool own = wv == 6u;
            const u32 t = wv % 3u, gw = B * 2u + wv / 3u;
            const u32 q = own ? (rank & 3u) : ((rank ^ (t + 1u)) & 3u);
            char* s0 = p.ws[q];
            char* s1 = p.ws[q | 4u];
            if (!own) {
                const rsrc_t r0 = mk_rsrc(s0 + OFF_PI + par * PBYTES, PBYTES), r1 = mk_rsrc(s1 + OFF_PI + par * PBYTES, PBYTES);
                const rsrc_t ru = mk_rsrc(me + OFF_UI, UIBYTES);
                for (u32 it = 0; it < (u32)IT1; ++it) {
                    const u32 n = gw + NW * it, dc = ORD1[n >> 2], b = n & 3u;
                    const u32 op = dc >> 6, half = op ? (rank & 1u) : ((rank >> 1) & 1u);
                    const u32 ci = ((op * 2u + half) * 16u + ((dc >> 2) & 15u)) * 8u + (rank >> 2) * 4u + (dc & 3u);
                    const u32 so = (ci * 4u + b) * 4096u + lane * 16u;
                    const u32x2 h = wait_slots(ctl + (wv * IT1 + it) * 2u);
                    if ((h[0] | h[1]) & H_POISON) continue;
                    const Blk k0 = load_block(r0, so, h[0], lane), k1 = load_block(r1, so, h[1], lane);
                    __builtin_amdgcn_sched_barrier(0);
                    const u32 uo = (dc * 6u + t * 2u) * 16384u + b * 4096u + lane * 16u;
                    u32 d[16];
                    unpack(k0, h[0], lane, d);
#pragma unroll
                    for (int u = 0; u < 4; ++u) __builtin_amdgcn_raw_buffer_store_b128((u32x4){d[4 * u], d[4 * u + 1], d[4 * u + 2], d[4 * u + 3]}, ru, uo + u * 1024u, 0, 17);
                    unpack(k1, h[1], lane, d);
#pragma unroll
                    for (int u = 0; u < 4; ++u) __builtin_amdgcn_raw_buffer_store_b128((u32x4){d[4 * u], d[4 * u + 1], d[4 * u + 2], d[4 * u + 3]}, ru, uo + 16384u + u * 1024u, 0, 17);
                    WAIT_VM(0);
                    if (lane == 0) __hip_atomic_store((GAS u32*)(me + OFF_RF + dc * 64u + (t * 4u + b) * 4u), call, __ATOMIC_RELAXED, __HIP_MEMORY_SCOPE_AGENT);
                    STAMP(2 + it);
                }
            }
            const rsrc_t r0 = mk_rsrc(s0 + OFF_PC + par * PBYTES, PBYTES), r1 = mk_rsrc(s1 + OFF_PC + par * PBYTES, PBYTES);
            const rsrc_t ro = mk_rsrc(p.out, IN_BYTES);
            const u32 nit = own ? ITO : IT2, n0 = own ? B : gw, nst = own ? NC : NW, sl = own ? SLO : SL2 + wv * IT2 * 2u;
            const u32 g4 = (u32)lane >> 4;
            const u32 lo = ((u32)lane & 15u) * 8192u + (((g4 & 1u) << 1) | (g4 >> 1)) * 16u;
            for (u32 it = 0; it < nit; ++it) {
                const u32 n = n0 + nst * it, lc = ORD2[n >> 1], hf = n & 1u, lvl = lc >> 4, C = lc & 15u;
                const u32 tile = (2u * lvl + (rank >> 2)) * 16u + C;
                const u32 so = (tile * 8u + (rank & 3u) * 2u + hf) * 4096u + lane * 16u;
                const u32 oo = (256u * (q >> 1) + 32u * lvl) * 8192u + (2048u * (q & 1u) + 128u * C + 64u * hf) * 2u + lo;
                const u32x2 h = wait_slots(ctl + sl + it * 2u);
                u32 o[16];
                if ((h[0] | h[1]) & H_POISON) {
#pragma unroll
                    for (int i = 0; i < 16; ++i) o[i] = 0x7fc07fc0u;
                } else {
                    const Blk k0 = load_block(r0, so, h[0], lane), k1 = load_block(r1, so, h[1], lane);
                    __builtin_amdgcn_sched_barrier(0);
                    u32 a[16], c[16];
                    unpack(k0, h[0], lane, a);
                    unpack(k1, h[1], lane, c);
#pragma unroll
                    for (int i = 0; i < 16; ++i) {
                        const float y0 = __uint_as_float(a[i] << 16) + __uint_as_float(c[i] << 16);
                        const float y1 = __uint_as_float(a[i] & 0xffff0000u) + __uint_as_float(c[i] & 0xffff0000u);
                        o[i] = cvt_pk(silu(y0), silu(y1));
                    }
                }
#pragma unroll
                for (int f = 0; f < 4; ++f) {
                    const auto sx = __builtin_amdgcn_permlane16_swap(o[4 * f], o[4 * f + 2], false, false);
                    const auto sy = __builtin_amdgcn_permlane16_swap(o[4 * f + 1], o[4 * f + 3], false, false);
                    __builtin_amdgcn_raw_buffer_store_b128((u32x4){sx[0], sy[0], sx[1], sy[1]}, ro, oo + (f >> 1) * 131072u + (f & 1) * 64u, 0, 0);
                }
                STAMP(12 + it);
            }
        }
        return;
    }

    const u32 wm = wv >> 1, wn = wv & 1u;
    const u32 g4 = (u32)lane >> 4, l15 = (u32)lane & 15u;
    u32 xa[2], wa[4][2];
#pragma unroll
    for (int ks = 0; ks < 2; ++ks) xa[ks] = (32u * wm + l15) * 128u + (((4u * ks + g4) ^ (l15 >> 1)) << 4);
#pragma unroll
    for (int ni = 0; ni < 4; ++ni)
#pragma unroll
        for (int t = 0; t < 2; ++t) {
            const u32 qq = l15 >> 2, pp = l15 & 3u, k = 8u * g4 + 4u * t + qq, ch = 8u * wn + 2u * ni + (pp >> 1);
            wa[ni][t] = LDS_W + 256u * k + ((ch ^ ((qq << 2) | ((k >> 2) & 3u))) << 4) + 8u * (pp & 1u);
        }
    const u32 lds0 = (u32)(size_t)smem + wv * 2048u;
    const rsrc_t rpc = mk_rsrc(me + OFF_PC + par * PBYTES, PBYTES);
    u32 tl = TILES[B - NC] | 256u;
    u32 poison = 0u;
    int sidx = 2;
    for (; tl & 256u; tl >>= 16) {
        const u32 R = (tl >> 4) & 15u, C = tl & 15u;
        u32x4 rx = mk_desc(p.x, IN_BYTES), rw = mk_desc(p.w, IN_BYTES);
        u32 vx0 = x_piece((rank >> 1) & 1u, R, 2u * wv, lane), vx1 = x_piece((rank >> 1) & 1u, R, 2u * wv + 1u, lane) - 1024u;
        u32 vw0 = w_piece(rank & 1u, C, 2u * wv, lane), vw1 = w_piece(rank & 1u, C, 2u * wv + 1u, lane) - 1024u;
        u32 sx = 0u, sw = 0u, dsx = 128u, dsw = 524288u;
        u32 ready = 0u;
        f32x4 acc[2][4];
#pragma unroll
        for (int mi = 0; mi < 2; ++mi)
#pragma unroll
            for (int ni = 0; ni < 4; ++ni) acc[mi][ni] = (f32x4){0.f, 0.f, 0.f, 0.f};

#define ISSUE(stage) do { u32 keep_; \
        asm volatile("s_mov_b32 %0, m0\n\ts_mov_b32 m0, %1\n\ts_nop 2\n\t" \
                     "buffer_load_dwordx4 %3, %7, %9 offen sc0 sc1 lds\n\t" \
                     "buffer_load_dwordx4 %4, %7, %9 offen offset:1024 sc0 sc1 lds\n\t" \
                     "s_mov_b32 m0, %2\n\ts_nop 0\n\t" \
                     "buffer_load_dwordx4 %5, %8, %10 offen sc0 sc1 lds\n\t" \
                     "buffer_load_dwordx4 %6, %8, %10 offen offset:1024 sc0 sc1 lds\n\t" \
                     "s_mov_b32 m0, %0" \
                     : "=&s"(keep_) : "s"(lds0 + (stage) * 16384u), "s"(lds0 + LDS_W + (stage) * 16384u), \
                       "v"(vx0), "v"(vx1), "v"(vw0), "v"(vw1), "s"(rx), "s"(rw), "s"(sx), "s"(sw) : "memory"); \
        sx += dsx; sw += dsw; } while (0)

        ISSUE(0); ISSUE(1); ISSUE(2);
        for (u32 it = 0; it < 8u; ++it) {
#pragma unroll
            for (int u = 0; u < 4; ++u) {
                const u32 is = it * 4u + u + 3u;
                bool poll = false;
                if (u & 1) poll = (ready < 4u && is == 8u + 6u * ready);
                if (poll && wv == 0u) {
                    const u32 kc = ((u32)lane >> 3) & 3u, op = ((u32)lane >> 2) & 1u, t = (u32)lane & 3u;
                    const u32 fo = (((op ? 16u + C : R) * 4u + kc) * 64u) + t * 16u;
                    const rsrc_t rf = mk_rsrc(me + OFF_RF, 8192u);
                    const u64 t0 = __builtin_amdgcn_s_memrealtime();
                    u32 nr;
                    for (;;) {
                        const u32x4 v = __builtin_amdgcn_raw_buffer_load_b128(rf, fo, 0, 17);
                        const u32 tv = __hip_atomic_load(tmo, __ATOMIC_RELAXED, __HIP_MEMORY_SCOPE_AGENT);
                        const bool ok = t == 3u || (v[0] == call && v[1] == call && v[2] == call && v[3] == call);
                        const u32 miss = ~(u32)__builtin_amdgcn_ballot_w64(ok);
                        nr = miss ? (u32)__builtin_ctz(miss) >> 3 : 4u;
                        if (nr > ready) break;
                        if (uni(tv) == call || __builtin_amdgcn_s_memrealtime() - t0 > DEADLINE) {
                            if (lane == 0) __hip_atomic_store(tmo, call, __ATOMIC_RELAXED, __HIP_MEMORY_SCOPE_AGENT);
                            nr = 4u | H_POISON;
                            break;
                        }
                        __builtin_amdgcn_s_sleep(4);
                        asm volatile("" ::: "memory");
                    }
                    if (lane == 0) ctl[0] = nr;
                }
                if (u < 2) { WAIT_VM(8); }
                else if (it < 7u) { WAIT_VM(8); }
                else if (u == 2) { WAIT_VM(4); }
                else { WAIT_VM(0); }
                asm volatile("s_waitcnt lgkmcnt(0)" ::: "memory");
                __builtin_amdgcn_s_barrier();
                asm volatile("" ::: "memory");
                if (poll) {
                    const u32 nr = uni(ctl[0]);
                    ready = nr & 7u; poison |= nr & H_POISON;
                    STAMP(sidx); ++sidx;
                }
                if (u == 1 && is == 8u) {
                    rx = mk_desc(me + OFF_UI, UIBYTES); rw = rx;
                    vx0 = vx1 = vw0 = vw1 = wv * 2048u + lane * 16u;
                    sx = R * 24u * 16384u; sw = (16u + C) * 24u * 16384u; dsx = dsw = 16384u;
                }
                if (u == 0 || it < 7u) ISSUE((u + 3) & 3);
#pragma unroll
                for (int ks = 0; ks < 2; ++ks) {
                    s16x8 xf[2], wf[4];
#pragma unroll
                    for (int mi = 0; mi < 2; ++mi) xf[mi] = *(LAS s16x8*)(smem + xa[ks] + u * 16384 + mi * 2048);
#pragma unroll
                    for (int ni = 0; ni < 4; ++ni) {
                        const s16x4 a = __builtin_amdgcn_ds_read_tr16_b64_v4i16((LAS s16x4*)(smem + wa[ni][0] + u * 16384 + ks * 8192));
                        const s16x4 b = __builtin_amdgcn_ds_read_tr16_b64_v4i16((LAS s16x4*)(smem + wa[ni][1] + u * 16384 + ks * 8192));
                        wf[ni] = __builtin_shufflevector(a, b, 0, 1, 2, 3, 4, 5, 6, 7);
                    }
#pragma unroll
                    for (int mi = 0; mi < 2; ++mi)
#pragma unroll
                        for (int ni = 0; ni < 4; ++ni) acc[mi][ni] = __builtin_amdgcn_mfma_f32_16x16x32_bf16(wf[ni], xf[mi], acc[mi][ni], 0, 0, 0);
                }
            }
        }
#undef ISSUE
        STAMP(sidx); ++sidx;
        u32 d[16];
#pragma unroll
        for (int mi = 0; mi < 2; ++mi)
#pragma unroll
            for (int ni = 0; ni < 4; ++ni) {
                d[(mi * 4 + ni) * 2] = cvt_pk(acc[mi][ni][0], acc[mi][ni][1]);
                d[(mi * 4 + ni) * 2 + 1] = cvt_pk(acc[mi][ni][2], acc[mi][ni][3]);
            }
        const u32 hdr = pack_store(d, rpc, ((R * 16u + C) * 8u + wv) * 4096u + lane * 16u, lane) | poison;
        char* q = p.ws[(4u * (R & 1u) + wm) & 7u];
        asm volatile("" : "+s"(q));
        WAIT_VM(0);
        if (lane == 0) {
            GAS u64* g = (GAS u64*)(q + OFF_H2 + ((par * 8u + rank) * 256u + ((R >> 1) * 16u + C) * 2u + wn) * 8u);
            __hip_atomic_store(g, ((u64)hdr << 32) | call, __ATOMIC_RELAXED, __HIP_MEMORY_SCOPE_SYSTEM);
        }
        STAMP(sidx); ++sidx;
    }
}

}

extern "C" void kernel_launch(int rank, int nranks, ncclComm_t comm, hipStream_t stream,
                              void* const* d_in, const int* in_sizes, int n_in,
                              void* d_out, int out_size, const int* in_dtypes, int out_dtype,
                              void* const* d_out_all, void* const* d_ws_all, size_t ws_bytes) {
    (void)comm; (void)in_dtypes; (void)out_dtype; (void)d_out_all;
    if (nranks != NR || (unsigned)rank >= (unsigned)NR || n_in != 2 || in_sizes[0] != 4096 * 512 || in_sizes[1] != 4096 * 512 ||
        out_size != 512 * 4096 || ws_bytes < WS_NEED) {
        fprintf(stderr, "kernel_launch: unexpected problem shape\n");
        return;
    }
    Params p;
    for (int i = 0; i < NR; ++i) p.ws[i] = (char*)d_ws_all[i];
    p.x = (const char*)d_in[0];
    p.w = (const char*)d_in[1];
    p.out = (char*)d_out;
    p.rank = rank;
    p.pad = 0;
    fused<<<NWG, 512, 0, stream>>>(p);
}
```

```cpp
#include <hip/hip_runtime.h>
#include <rccl/rccl.h>
#include <cstdio>

typedef unsigned u32;
typedef unsigned long long u64;
typedef u32 u32x2 __attribute__((ext_vector_type(2)));
typedef u32 u32x3 __attribute__((ext_vector_type(3)));
typedef u32 u32x4 __attribute__((ext_vector_type(4)));
typedef u32 u32x8 __attribute__((ext_vector_type(8)));
typedef unsigned short u16x2 __attribute__((ext_vector_type(2)));
typedef short s16x4 __attribute__((ext_vector_type(4)));
typedef short s16x8 __attribute__((ext_vector_type(8)));
typedef float f32x2 __attribute__((ext_vector_type(2)));
typedef float f32x4 __attribute__((ext_vector_type(4)));
typedef __bf16 bf16x2 __attribute__((ext_vector_type(2)));
typedef __amdgpu_buffer_rsrc_t rsrc_t;
#define LAS __attribute__((address_space(3)))
#define GAS __attribute__((address_space(1)))

namespace {

constexpr int NR = 8;
constexpr u32 NWG = 256;
constexpr u32 NC = 36;
constexpr u32 NG = NC * 2 / 3;
constexpr u32 GW = 3;
constexpr u32 NOWN = 12;
constexpr u32 NPP = 6;
constexpr u32 PB0 = 84, PW = 2;

constexpr u32 OFF_CNT = 0;
constexpr u32 OFF_TMO = 16u << 10;
constexpr u32 OFF_TK = 20u << 10;
constexpr u32 PBYTES = (16u << 20) + (256u << 10);
constexpr u32 OFF_PK = 1u << 20;
constexpr u32 OFF_UI = OFF_PK + 2 * PBYTES;
constexpr u32 UIBYTES = 12u << 20;
constexpr u32 OFF_P3 = OFF_UI + 2 * UIBYTES;
constexpr u32 P3BYTES = 3u << 20;
constexpr u32 HS = 512, H1S = 256;
constexpr u32 OFF_H = OFF_P3 + P3BYTES, H2_OFF = 512u * H1S, HBYTES = H2_OFF + 512u * HS;
constexpr u32 REP = 16;
constexpr u32 OFF_RF = OFF_H + 2u * HBYTES, RFS = REP * 256u;
constexpr u32 WS_NEED = OFF_RF + 129u * RFS;
constexpr u32 IN_BYTES = 4u << 20;
constexpr u32 OOB = 0x7ffffff0u;
constexpr u64 DEADLINE = 100000000ull;
constexpr u32 H_ODD = 1u << 23, H_DENSE = 1u << 24, H_POISON = 1u << 30;

constexpr int LDS_W = 65536, LDS_CTL = 131072, LDS_HDR = LDS_CTL + 256, LDS_ENC = LDS_HDR + 512, LDS_DEC = LDS_ENC + 1536, LDS_BYTES = LDS_DEC + 4096;

__device__ constexpr unsigned ORD1[128] = {0,4,8,12,64,68,72,76,16,20,24,28,80,84,1,5,9,13,65,69,73,77,17,21,25,29,81,85,2,6,10,14,66,70,74,78,18,22,26,30,32,36,40,44,82,86,3,7,11,15,67,71,75,88,79,19,23,27,31,92,33,37,41,45,83,48,52,56,60,87,89,93,96,34,38,42,46,49,53,57,61,100,90,104,94,97,35,39,43,47,108,50,54,58,62,101,112,91,105,116,95,98,109,51,55,59,63,120,102,113,124,106,117,99,110,121,103,114,125,107,118,111,122,115,126,119,123,127};
__device__ constexpr unsigned INV1[128] = {0,14,28,46,1,15,29,47,2,16,30,48,3,17,31,49,8,22,36,55,9,23,37,56,10,24,38,57,11,25,39,58,40,60,73,86,41,61,74,87,42,62,75,88,43,63,76,89,65,77,91,103,66,78,92,104,67,79,93,105,68,80,94,106,4,18,32,50,5,19,33,51,6,20,34,52,7,21,35,54,12,26,44,64,13,27,45,69,53,70,82,97,59,71,84,100,72,85,101,113,81,95,108,116,83,98,111,119,90,102,114,121,96,109,117,123,99,112,120,125,107,115,122,126,110,118,124,127};
__device__ constexpr unsigned ORD2[64] = {0,1,2,3,16,17,18,19,4,20,5,21,32,33,34,35,36,37,6,22,38,7,23,39,48,49,50,51,52,53,54,55,8,24,40,56,9,25,41,57,10,26,42,58,11,27,43,59,12,28,44,60,13,29,45,61,14,30,46,62,15,31,47,63};
__device__ constexpr unsigned INV2[64] = {0,1,2,3,8,10,18,21,32,36,40,44,48,52,56,60,4,5,6,7,9,11,19,22,33,37,41,45,49,53,57,61,12,13,14,15,16,17,20,23,34,38,42,46,50,54,58,62,24,25,26,27,28,29,30,31,35,39,43,47,51,55,59,63};
__device__ constexpr unsigned TILES[220] = {17760513u,18809105u,19857697u,20906289u,30146816u,31195408u,32244000u,33292592u,21954819u,23003411u,24052003u,25100595u,17694978u,18743570u,19792162u,20840754u,26149185u,27197777u,28246369u,29294961u,21889344u,22937936u,23986528u,25035120u,30343491u,31392083u,32440675u,33489267u,26083650u,27132242u,28180834u,29229426u,33816837u,33882389u,33947941u,34013493u,30277892u,31326484u,32375076u,33423668u,34079045u,34144597u,34210149u,34275701u,33554756u,33620308u,33685860u,33751412u,385u,401u,417u,433u,384u,400u,416u,432u,387u,403u,419u,435u,386u,402u,418u,434u,389u,405u,421u,437u,388u,404u,420u,436u,263u,279u,295u,311u,262u,278u,294u,310u,327u,343u,359u,375u,326u,342u,358u,374u,391u,407u,423u,439u,390u,406u,422u,438u,449u,465u,481u,497u,448u,464u,480u,496u,451u,467u,483u,499u,450u,466u,482u,498u,453u,469u,485u,501u,452u,468u,484u,32769280u,455u,471u,487u,503u,454u,30803216u,31851808u,32900400u,17368321u,18416913u,19465505u,20514097u,17302848u,18351440u,19399938u,20448530u,21562689u,22611281u,23659779u,24708371u,21497122u,22545714u,23594306u,24642898u,25756963u,26805555u,27854147u,28902739u,25691396u,26739988u,27788580u,28837172u,29951237u,30999829u,32048421u,33097013u,29885764u,30934356u,31982854u,33031446u,17499461u,18548053u,19596551u,20645143u,17433894u,18482486u,19531078u,20579670u,21693735u,22742327u,23790919u,24839511u,21628168u,22676760u,23725352u,24773944u,25888009u,26936601u,27985193u,29033785u,25822536u,26871128u,27919626u,28968218u,30082377u,31130969u,32179467u,33228059u,30016810u,31065402u,32113994u,33162586u,17630507u,18679099u,19727691u,20776283u,17564940u,18613532u,19662124u,20710716u,21824781u,22873373u,23921965u,24970557u,21759308u,22807900u,23856398u,24904990u,26019149u,27067741u,28116239u,29164831u,25953582u,27002174u,28050766u,29099358u,30213423u,31262015u,32310607u,33359199u};

struct Tab { unsigned v[256]; constexpr const unsigned& operator[](unsigned i) const { return v[i]; } };
constexpr Tab merged() { Tab t{}; for (int i = 0; i < 192; ++i) t.v[i] = i < 128 ? ORD1[i] : ORD2[i - 128]; return t; }
__device__ constexpr Tab ORDS = merged();

static_assert(NPP % 2 == 0 && NC % 3 == 0 && PB0 >= NC && (NWG - PB0) * PW >= NPP * 24 && sizeof(TILES) == (NWG - NC) * 4, "");

struct Params {
    char* ws[NR];
    const char* x;
    const char* w;
    char* out;
    char* me;
    int rank;
    int pad;
};

#define DEVI __device__ __forceinline__
#define WAIT_VM(n) asm volatile("s_waitcnt vmcnt(" #n ")" ::: "memory")

DEVI rsrc_t mk_rsrc(const void* p, u32 bytes) { return __builtin_amdgcn_make_buffer_rsrc((void*)p, 0, bytes, 0x00020000); }
DEVI u32x4 mk_desc(const void* p, u32 bytes) {
    const u64 a = (u64)p;
    return (u32x4){(u32)a, (u32)(a >> 32) & 0xffffu, bytes, 0x00020000u};
}
DEVI u32 uni(u32 v) { return __builtin_amdgcn_readfirstlane(v); }
DEVI u32 bfi(u32 m, u32 a, u32 b) { return __builtin_amdgcn_bitop3_b32(m, a, b, 0xca); }
DEVI u32 cvt_pk(float lo, float hi) {
    const bf16x2 r = __builtin_convertvector((f32x2){lo, hi}, bf16x2);
    return __builtin_bit_cast(u32, r);
}
DEVI u32 umax(u32 a, u32 b) { return a > b ? a : b; }
DEVI u32 wave_max(u32 v) {
    v = umax(v, (u32)__builtin_amdgcn_update_dpp(0, (int)v, 0x111, 0xf, 0xf, false));
    v = umax(v, (u32)__builtin_amdgcn_update_dpp(0, (int)v, 0x112, 0xf, 0xf, false));
    v = umax(v, (u32)__builtin_amdgcn_update_dpp(0, (int)v, 0x114, 0xf, 0xf, false));
    v = umax(v, (u32)__builtin_amdgcn_update_dpp(0, (int)v, 0x118, 0xf, 0xf, false));
    v = umax(v, (u32)__builtin_amdgcn_update_dpp(0, (int)v, 0x142, 0xa, 0xf, false));
    v = umax(v, (u32)__builtin_amdgcn_update_dpp(0, (int)v, 0x143, 0xc, 0xf, false));
    return (u32)__builtin_amdgcn_readlane((int)v, 63);
}

DEVI u32 wave_scan(u32 v) {
    v += (u32)__builtin_amdgcn_update_dpp(0, (int)v, 0x111, 0xf, 0xf, false);
    v += (u32)__builtin_amdgcn_update_dpp(0, (int)v, 0x112, 0xf, 0xf, false);
    v += (u32)__builtin_amdgcn_update_dpp(0, (int)v, 0x114, 0xf, 0xf, false);
    v += (u32)__builtin_amdgcn_update_dpp(0, (int)v, 0x118, 0xf, 0xf, false);
    v += (u32)__builtin_amdgcn_update_dpp(0, (int)v, 0x142, 0xa, 0xf, false);
    v += (u32)__builtin_amdgcn_update_dpp(0, (int)v, 0x143, 0xc, 0xf, false);
    return v;
}


DEVI u32 slot_off(u32 i) { return i * 3072u; }
DEVI u32 exc_off(u32 i) { return (12u << 20) + i * 1024u; }
DEVI u32 list_off(u32 i) { return (16u << 20) + i * 64u; }
DEVI u32 window_of(u32 top) {
    const u32 m = wave_max(top), odd = m < 200u ? ~m & 1u : 0u, t = (m + odd) >> 1;
    return ((t > 7u ? t - 7u : 0u) << 16) | (odd << 23);
}
DEVI void swap_planes(u32& a0, u32& a1, u32& a2, u32& a3) {
    const u32 e = bfi(0x11111111u, a0 >> 2, bfi(0x22222222u, a1 >> 1, bfi(0x88888888u, a2 << 1, a3)));
    a0 = bfi(0x44444444u, a3 << 2, a0);
    a1 = bfi(0x44444444u, a3 << 1, a1);
    a2 = bfi(0x44444444u, a3 >> 1, a2);
    a3 = e;
}
DEVI void pack_low(const u32 (&d)[16], rsrc_t r, u32 slot, int lane) {
    u32 L[8];
#pragma unroll
    for (int i = 0; i < 8; ++i) L[i] = __builtin_amdgcn_perm(d[2 * i + 1], d[2 * i], 0x06040200u);
    __builtin_amdgcn_raw_buffer_store_b128((u32x4){L[0], L[1], L[2], L[3]}, r, slot_off(slot) + lane * 16u, 0, 17);
    __builtin_amdgcn_raw_buffer_store_b128((u32x4){L[4], L[5], L[6], L[7]}, r, slot_off(slot) + 1024u + lane * 16u, 0, 17);
}
struct High { u32 a[4]; u32x4 b; };
DEVI High split_high(const u32 (&d)[16], u32 hdr) {
    const u32 brep = ((hdr >> 16) & 127u) * 0x01010101u;
    High h;
#pragma unroll
    for (int i = 0; i < 4; ++i) {
        const u32 he = __builtin_amdgcn_perm(d[4 * i + 1], d[4 * i], 0x07050301u) - brep;
        const u32 ho = __builtin_amdgcn_perm(d[4 * i + 3], d[4 * i + 2], 0x07050301u) - brep, ro = __builtin_amdgcn_alignbit(ho, ho, 28);
        h.a[i] = bfi(0x87878787u, he, ro);
        h.b[i] = bfi(0x87878787u, ro, he);
    }
    swap_planes(h.a[0], h.a[1], h.a[2], h.a[3]);
    return h;
}
DEVI void pack_high(const High& h, u32& hdr, rsrc_t r, u32 slot, int lane) {
    __builtin_amdgcn_raw_buffer_store_b96((u32x3){h.a[0], h.a[1], h.a[2]}, r, slot_off(slot) + 2048u + lane * 12u, 0, 17);
    const u64 bal = __builtin_amdgcn_ballot_w64(((h.b[0] | h.b[1] | h.b[2]) | h.b[3]) != 0u);
    u64 quad;
    asm("s_quadmask_b64 %0, %1" : "=s"(quad) : "s"(bal) : "scc");
    if (quad) __builtin_amdgcn_raw_buffer_store_b128(h.b, r, (((u32)quad >> (lane >> 2)) & 1u) ? exc_off(slot) + lane * 16u : OOB, 0, 17);
    hdr |= (u32)quad;
}
template <int NB> DEVI void pack_planes(const u32 (&E)[NB], u32 (&hdr)[NB], rsrc_t r, const u32 (&slot)[NB], LAS char* scr, int lane, u64 xp) {
    if (!(hdr[0] & hdr[NB - 1] & H_DENSE)) {
        u32 N[NB], at[NB], cnt = 0u;
#pragma unroll
        for (int j = 0; j < NB; ++j) { N[j] = ~E[j]; cnt |= (u32)__builtin_popcount(N[j]) << (16 * j); }
        const u32 incl = wave_scan(cnt), tot = (u32)__builtin_amdgcn_readlane((int)incl, 63), tag = ((u32)lane & 7u) << 5;
        const u32 sc = (u32)(size_t)scr, dump = sc + 128u + ((u32)lane & 15u) * 4u;
#pragma unroll
        for (int j = 0; j < NB; ++j) {
            if (((tot >> (16 * j)) & 0xffffu) > 56u) hdr[j] |= H_DENSE;
            if (hdr[j] & H_DENSE) N[j] = 0u;
            *(volatile LAS unsigned char*)(size_t)((lane & 7) == 7 ? sc + 64u * j + ((u32)lane >> 3) : dump) = (unsigned char)(incl >> (16 * j));
            at[j] = sc + 64u * j + 8u + (((incl - cnt) >> (16 * j)) & 0xffffu);
        }
        do {
#pragma unroll
            for (int rep = 0; rep < 2; ++rep)
#pragma unroll
                for (int j = 0; j < NB; ++j) {
                    *(volatile LAS unsigned char*)(size_t)(N[j] ? at[j] : dump) = (unsigned char)(tag | (u32)__builtin_ctz(N[j] | 0x80000000u));
                    N[j] &= N[j] - 1u;
                    ++at[j];
                }
        } while (__builtin_amdgcn_ballot_w64((N[0] | N[NB - 1]) != 0u) != 0ull);
        const u32 X = *(volatile LAS u32*)(scr + (lane & 31) * 4);
        const bool second = NB == 2 && lane >= 16;
        const bool put = lane < 16 * NB && !((second ? hdr[NB - 1] : hdr[0]) & H_DENSE);
        if (NB == 1) __builtin_amdgcn_raw_buffer_store_b32(X, r, put ? list_off(slot[0]) + lane * 4u : OOB, 0, 17);
        else if (put) __hip_atomic_store((GAS u32*)xp, X, __ATOMIC_RELAXED, __HIP_MEMORY_SCOPE_SYSTEM);
    }
#pragma unroll
    for (int j = 0; j < NB; ++j)
        if (hdr[j] & H_DENSE) __builtin_amdgcn_raw_buffer_store_b32(E[j], r, slot_off(slot[j]) + 2816u + lane * 4u, 0, 17);
}

struct Blk { u32x4 l0, l1, b; u32x3 p; u32 x; };
DEVI u32 plane_of(u32 x, int l0, LAS char* scr, int lane) {
    const u32 c0 = (u32)__builtin_amdgcn_readlane((int)x, l0), c1 = (u32)__builtin_amdgcn_readlane((int)x, l0 + 1);
    const u32 by = (u32)__builtin_amdgcn_ds_bpermute(((lane + 8) & ~3) + l0 * 4, (int)x) >> (((u32)lane & 3u) * 8u);
    const u32 lr = (u32)lane * 0x01010101u | 0x80808080u;
    const u32 seg = __builtin_popcount((lr - c0) & 0x80808080u) + __builtin_popcount((lr - c1) & 0x80808080u);
    __hip_atomic_fetch_or((LAS u32*)(scr + (((seg << 5) | ((by >> 3) & 28u)) & 255u)), (u32)lane < (c1 >> 24) ? 1u << (by & 31u) : 0u,
                          __ATOMIC_RELAXED, __HIP_MEMORY_SCOPE_WAVEFRONT);
    return ~__hip_atomic_exchange((LAS u32*)(scr + lane * 4), 0u, __ATOMIC_RELAXED, __HIP_MEMORY_SCOPE_WAVEFRONT);
}
DEVI void high_bytes(const Blk& k, u32 plane, u32 hdr, u32 (&H)[8]) {
    const u32 brep = ((hdr >> 16) & 127u) * 0x01010101u;
    u32 A[4] = {k.p[0], k.p[1], k.p[2], plane};
    swap_planes(A[0], A[1], A[2], A[3]);
#pragma unroll
    for (int i = 0; i < 4; ++i) {
        const u32 t = bfi(0x87878787u, k.b[i], A[i]);
        H[2 * i] = bfi(0x87878787u, A[i], k.b[i]) + brep;
        H[2 * i + 1] = __builtin_amdgcn_alignbit(t, t, 4) + brep;
    }
}
DEVI u32 low_bytes(const Blk& k, int i) { return i < 4 ? k.l0[i] : k.l1[i - 4]; }
DEVI u32x4 as_bf16(const Blk& k, const u32 (&H)[8], int u) {
    return (u32x4){__builtin_amdgcn_perm(H[2 * u], low_bytes(k, 2 * u), 0x05010400u), __builtin_amdgcn_perm(H[2 * u], low_bytes(k, 2 * u), 0x07030602u),
                   __builtin_amdgcn_perm(H[2 * u + 1], low_bytes(k, 2 * u + 1), 0x05010400u), __builtin_amdgcn_perm(H[2 * u + 1], low_bytes(k, 2 * u + 1), 0x07030602u)};
}
DEVI f32x2 as_f32(const Blk& k, const u32 (&H)[8], int j) {
    const u32 s0 = (j & 1) ? 0x06020c0cu : 0x04000c0cu;
    return (f32x2){__uint_as_float(__builtin_amdgcn_perm(H[j >> 1], low_bytes(k, j >> 1), s0)),
                   __uint_as_float(__builtin_amdgcn_perm(H[j >> 1], low_bytes(k, j >> 1), s0 + 0x01010000u))};
}

DEVI f32x2 silu2(f32x2 y) {
    constexpr float L2E = 1.44269504088896340736f;
    const f32x2 z = y * -L2E;
    f32x2 e;
    e.x = __builtin_amdgcn_exp2f(__builtin_amdgcn_fmed3f(z.x, -60.0f * L2E, 60.0f * L2E));
    e.y = __builtin_amdgcn_exp2f(__builtin_amdgcn_fmed3f(z.y, -60.0f * L2E, 60.0f * L2E));
    e += 1.0f;
    return y * (f32x2){__builtin_amdgcn_rcpf(e.x), __builtin_amdgcn_rcpf(e.y)};
}

DEVI u32 x_row0(u32 half, u32 panel) { return (panel & 3u) * 1024u + 256u * half + 64u * (panel >> 2); }
DEVI u32 x_piece(u32 row0, u32 gs, u32 p, int lane) {
    const u32 r = p * 8u + ((u32)lane >> 3);
    return (row0 + (r >> 6) * gs + (r & 63u)) * 1024u + ((((u32)lane & 7u) ^ ((r >> 1) & 7u)) << 4);
}
DEVI u32 w_piece(u32 half, u32 panel, u32 p, int lane) {
    const u32 k = p * 4u + ((u32)lane >> 4);
    return k * 8192u + (2048u * half + 128u * panel) * 2u + ((((u32)lane & 15u) ^ (((k & 3u) << 2) | ((k >> 2) & 3u))) << 4);
}

__global__ void __launch_bounds__(512, 2) fused(Params p) {
    __shared__ __attribute__((aligned(16))) char smem_[LDS_BYTES];
    LAS char* smem = (LAS char*)smem_;
    const int lane = threadIdx.x & 63;
    const u32 wv = uni(threadIdx.x >> 6);
    const u32 B = blockIdx.x;
    const u64 tstart = __builtin_amdgcn_s_memrealtime();
    u32 J = B - NC;
    u32 cpos = ((J >> 2) * 2u + (wv >> 2) + NPP) & 127u;
    const u32 pj = (B - PB0) * PW + wv;
    u32 dc = ORD1[cpos], dcp = ORD1[(pj / 24u) & 127u], rank_ = (u32)p.rank;
    const char* px_ = p.x;
    const char* pw_ = p.w;
    char* me = p.me;
    u32 lx[4], lw[4];
#pragma unroll
    for (int u = 0; u < 4; ++u) {
        lx[u] = x_piece(0u, 0u, u, lane); lw[u] = w_piece(0u, 0u, u, lane);
        asm volatile("" : "+v"(lx[u]), "+v"(lw[u]));
    }
    asm volatile("" : "+s"(dc), "+s"(dcp), "+s"(rank_), "+s"(px_), "+s"(pw_), "+s"(me));
    auto fetch = [&](u32x4 (&v)[4], u32 e, u32 half, u32 h, u32 b, u32 bytes) {
        const u32 op = e >> 6, panel = (e >> 2) & 15u, kk = h * 4u + (e & 3u);
        const rsrc_t rs = mk_rsrc(op ? pw_ : px_, bytes);
        const u32 so = op ? (kk * 64u + b * 16u) * 8192u + (2048u * half + 128u * panel) * 2u : (x_row0(half, panel) + (b >> 1) * 512u + (b & 1u) * 32u) * 1024u + kk * 128u;
#pragma unroll
        for (int u = 0; u < 4; ++u) v[u] = __builtin_amdgcn_raw_buffer_load_b128(rs, bfi(0u - op, lw[u], lx[u]), so, 0);
    };
    const u32 rank = rank_ & 7u;
    GAS u32* tmo = (GAS u32*)(me + OFF_TMO);
    LAS u32* ctl = (LAS u32*)(smem + LDS_CTL);
    GAS u32* cnt = (GAS u32*)(me + OFF_CNT + B * 64u + wv * 4u);

    const u32 cnt_v = __hip_atomic_load(cnt, __ATOMIC_RELAXED, __HIP_MEMORY_SCOPE_AGENT);
    if (threadIdx.x < 16) *(LAS u32x4*)(ctl + threadIdx.x * 4) = (u32x4){0u, 0u, 0u, 0u};
    if (B < NC) {
        asm volatile("s_waitcnt lgkmcnt(0)" ::: "memory");
        __builtin_amdgcn_s_barrier();
        asm volatile("" ::: "memory");
    }
    if (B >= PB0 && wv < PW && pj < NPP * 24u) {
        const u32 b = pj & 3u, h = (pj >> 2) & 1u, te = (pj >> 3) % 3u + 1u, dst = (h << 2) | ((rank ^ te) & 3u);
        const bool own_rows = dcp < 64u && ((dcp >> 1) & 6u) + (b >> 1) == rank;
        u32x4 v[4];
        fetch(v, dcp, (dst >> (1u - (dcp >> 6))) & 1u, h, b, own_rows ? 0u : IN_BYTES);
        __builtin_amdgcn_sched_barrier(0);
        const u32 c1 = uni(cnt_v) + 1u;
        char* qd = p.ws[dst & 7u];
        const rsrc_t ru = mk_rsrc(qd + OFF_UI + (c1 & 1u) * UIBYTES, UIBYTES);
        const u32 uo = (dcp * 6u + (te - 1u) * 2u + (rank >> 2)) * 16384u + b * 4096u + lane * 16u;
#pragma unroll
        for (int u = 0; u < 4; ++u) __builtin_amdgcn_raw_buffer_store_b128(v[u], ru, uo + u * 1024u, 0, 17);
        WAIT_VM(0);
        if (lane == 0)
            __hip_atomic_store((GAS unsigned short*)(qd + OFF_RF + dcp * RFS + (c1 & 1u) * 128u + ((b & 2u) * 4u + (te - 1u) * 2u + (rank >> 2)) * 4u + (b & 1u) * 2u),
                               (unsigned short)(c1 | 0x8000u), __ATOMIC_RELAXED, __HIP_MEMORY_SCOPE_SYSTEM);
    }

    constexpr u32 NJ = (128u - NPP) * 2u, NX = NJ - (NWG - NC);
    u32x4 v[4];
    if (J < NJ) {
        if (J >= 36u) while (__builtin_amdgcn_s_memrealtime() - tstart < 32u + J * 3u) __builtin_amdgcn_s_sleep(2);
        fetch(v, dc, J & 1u, (J >> 1) & 1u, wv & 3u, IN_BYTES);
    }
    __builtin_amdgcn_sched_barrier(0);
    for (u32 hb = LDS_HDR; J < NJ; hb += 256u) {
        const u32 half = J & 1u, h = (J >> 1) & 1u, b = wv & 3u, op = dc >> 6, panel = (dc >> 2) & 15u;
        u32 d[16];
#pragma unroll
        for (int i = 0; i < 16; ++i) d[i] = v[i >> 2][i & 3];
        const u32 d0 = (h << 2) | (op ? half : half << 1), d1 = d0 | (op ? 2u : 1u);
        char* q0 = p.ws[d0 & 7u];
        char* q1 = p.ws[d1 & 7u];
        const u32 call = uni(cnt_v) + 1u, par = call & 1u;
        const u32 ci = (half * 2u + h) * 128u + cpos;
        u16x2 mx = __builtin_bit_cast(u16x2, d[0] & 0x7fff7fffu);
#pragma unroll
        for (int i = 1; i < 16; ++i) mx = __builtin_elementwise_max(mx, __builtin_bit_cast(u16x2, d[i] & 0x7fff7fffu));
        const u32 win = window_of((u32)(mx.x > mx.y ? mx.x : mx.y) >> 7), up = ((win >> 23) & 1u) * 0x00800080u;
#pragma unroll
        for (int i = 0; i < 16; ++i) d[i] += up;
        const rsrc_t rp = mk_rsrc(me + OFF_PK + par * PBYTES, PBYTES);
        u32 hd[1] = {win | (J < 12u ? H_DENSE : 0u)};
        const u32 sl[1] = {ci * 4u + b};
        pack_low(d, rp, sl[0], lane);
        const High hi = split_high(d, win);
        pack_high(hi, hd[0], rp, sl[0], lane);
        const u32 pl[1] = {hi.a[3]};
        pack_planes<1>(pl, hd, rp, sl, smem + LDS_ENC + wv * 192u, lane, 0ull);
        const u32 hdr = hd[0];
        if (lane < 2) {
            const u32 dst = lane ? d1 : d0;
            const bool own_rows = op == 0u && 2u * (panel & 3u) + (b >> 1) == rank;
            const u64 g = (dst & 3u) != (rank & 3u) && !own_rows ? (u64)((lane ? q1 : q0) + OFF_H + par * HBYTES + (rank * 64u + (cpos >> 1)) * H1S + wv * 8u) : 0ull;
            *(LAS u32x4*)(smem + hb + (lane * 8u + wv) * 16u) = (u32x4){(u32)g, (u32)(g >> 32), call, hdr};
        }
        WAIT_VM(0);
        asm volatile("s_waitcnt lgkmcnt(0)" ::: "memory");
        __builtin_amdgcn_s_barrier();
        asm volatile("" ::: "memory");
        if (wv == 7u && lane < 16) {
            const u32x4 e = *(LAS u32x4*)(smem + hb + lane * 16u);
            const u64 g = ((u64)e[1] << 32) | e[0];
            if (g) __hip_atomic_store((GAS u64*)g, ((u64)e[3] << 32) | e[2], __ATOMIC_RELAXED, __HIP_MEMORY_SCOPE_SYSTEM);
        }
        J = J >= NWG - NC - NX ? J + NX : NJ;
        cpos = ((J >> 2) * 2u + (wv >> 2) + NPP) & 127u;
        dc = ORD1[cpos];
        if (J < NJ) fetch(v, dc, J & 1u, (J >> 1) & 1u, wv & 3u, IN_BYTES);
    }
    const u32 call = uni(cnt_v) + 1u, par = call & 1u;
    if (lane == 0) __hip_atomic_store(cnt, call, __ATOMIC_RELAXED, __HIP_MEMORY_SCOPE_AGENT);

    const u32 rtag = ((call & 0x7fffu) | 0x8000u) * 0x10001u;
    u32 n;
    if (B < NC) {
        const u32 gi = wv >> 1;
        if (gi >= GW) return;
        n = NPP * 4u + gi * NG + (B / 3u) * 2u + (wv & 1u);
    } else {
    u32 tl = TILES[B - NC];
    if (wv >= 4u) {
        __builtin_amdgcn_s_setprio(3);
        const u32 op = (wv >> 1) & 1u, hf = wv & 1u;
        const u32 lds0 = (u32)(size_t)smem + op * LDS_W + hf * 8192u;
        const u32x4 rui = mk_desc(me + OFF_UI + par * UIBYTES, UIBYTES);
        const u32 vr = hf * 8192u + lane * 16u;
        for (; tl & 256u; tl >>= 16) {
            const u32 px = (tl >> 4) & 15u, panel = op ? 16u + (tl & 15u) : px;
            const bool brief = tl & 1024u;
#define DMA4(S) "buffer_load_dwordx4 %2, %6, " S " offen sc0 sc1 lds\n\t" \
                "buffer_load_dwordx4 %3, %6, " S " offen offset:1024 sc0 sc1 lds\n\t" \
                "buffer_load_dwordx4 %4, %6, " S " offen offset:2048 sc0 sc1 lds\n\t" \
                "buffer_load_dwordx4 %5, %6, " S " offen offset:3072 sc0 sc1 lds\n\t"
#define ISSUE(step, v0, v1, v2, v3, rd, so, dq) do { u32 keep_; \
            asm volatile("s_mov_b32 %0, m0\n\ts_mov_b32 m0, %1\n\ts_nop 0\n\t" DMA4("%7") \
                         "s_add_u32 m0, m0, 0x1000\n\ts_nop 0\n\t" DMA4("%8") \
                         "s_mov_b32 m0, %0" \
                         : "=&s"(keep_) : "s"(lds0 + ((step) & 3u) * 16384u), "v"(v0), "v"(v1), "v"(v2), "v"(v3), \
                           "s"(rd), "s"(so), "s"((so) + (dq)) : "memory", "scc"); } while (0)
#define BARRIER() do { __builtin_amdgcn_s_barrier(); asm volatile("" ::: "memory"); } while (0)
            {
                const u32x4 rd = mk_desc(op ? p.w : p.x, IN_BYTES);
                const u32 qs = brief ? rank ^ ((px >> 1) + 1u) : rank, gs = brief ? 64u : 512u;
                const u32 row0 = brief ? rank * 512u + (qs & 2u) * 128u + (px & 1u) * 128u : x_row0((rank >> 1) & 1u, px);
                u32 vo[4];
#pragma unroll
                for (int u = 0; u < 4; ++u) vo[u] = (op ? w_piece(qs & 1u, panel & 15u, hf * 8u + u, lane) : x_piece(row0, gs, hf * 8u + u, lane)) - u * 1024u;
                const u32 dso = op ? 524288u : 128u, dq = op ? 131072u : 32768u;
#pragma unroll 1
                for (u32 c = 0; c < 8u; ++c) {
                    if (c >= 3u) { WAIT_VM(16); BARRIER(); }
                    ISSUE(c, vo[0], vo[1], vo[2], vo[3], rd, c * dso, dq);
                }
            }
            const u32 cp = ((op ? px : tl) & (REP - 1u)) * 256u, sb = (panel * 24u - 8u) * 16384u;
            u32 np = 0u;
#pragma unroll
            for (int kc = 0; kc < 4; ++kc) np += INV1[panel * 4u + kc] < NPP;
            np = uni(np);
            const char* rf = me + OFF_RF + par * 128u + hf * 32u + panel * 4u * RFS + (np ? 0u : cp);
            u32 issued = 8u, avail = 8u, seg = 8u;
            u32 i = brief ? 29u : 5u;
#define SEEN(miss) do { const u32 l_ = (u32)__builtin_ctz(uni(miss) | 64u); \
            avail = seg + l_; if (l_ == 6u) { seg += 6u; rf += --np ? RFS : RFS + cp; } } while (0)
#define ASK() u32x8 f; asm volatile("s_load_dwordx8 %0, %1, 0x0 glc" : "=&s"(f) : "s"(rf) : "memory")
#define THERE() do { asm volatile("s_waitcnt lgkmcnt(0)" : "+s"(f) :: "memory"); \
            u32 miss_ = 0u; \
            _Pragma("unroll") for (int j = 0; j < 6; ++j) { const u32 x_ = f[j] ^ rtag; miss_ |= ((x_ | (0u - x_)) >> 31) << j; } \
            SEEN(miss_); } while (0)
#define ISSUE_R() do { ISSUE(issued, vr, vr, vr, vr, rui, sb + issued * 16384u, 4096u); ++issued; } while (0)
#define STEP() do { WAIT_VM(16); BARRIER(); ISSUE_R(); ++i; } while (0)
#pragma unroll 1
            while (i < 29u || (issued < 32u && !brief)) {
                if (issued == i) {
                    const u32 soon = 380u + INV1[panel * 4u + (i - 8u) / 6u] * 17u;
                    while (__builtin_amdgcn_s_memrealtime() - tstart < soon) __builtin_amdgcn_s_sleep(8);
                    const u64 t0 = __builtin_amdgcn_s_memrealtime();
                    for (u32 c = 1;; ++c) {
                        if (c & 63u) {
                            ASK();
                            THERE();
                        } else {
                            const u32 v = __hip_atomic_load((GAS u32*)rf + (lane < 6 ? lane : 0), __ATOMIC_RELAXED, __HIP_MEMORY_SCOPE_SYSTEM);
                            SEEN((u32)__builtin_amdgcn_ballot_w64(v != rtag) & 63u);
                            if (uni(__hip_atomic_load(tmo, __ATOMIC_RELAXED, __HIP_MEMORY_SCOPE_AGENT)) == call || __builtin_amdgcn_s_memrealtime() - t0 > DEADLINE) {
                                if (lane == 0) { __hip_atomic_store(tmo, call, __ATOMIC_RELAXED, __HIP_MEMORY_SCOPE_AGENT); *(volatile LAS u32*)ctl = H_POISON; }
                                asm volatile("s_waitcnt lgkmcnt(0)" ::: "memory");
                                avail = seg = 32u;
                            }
                        }
                        if (avail > i) break;
                        __builtin_amdgcn_s_sleep(4);
                    }
                    __builtin_amdgcn_s_waitcnt(0xC07F);
                    asm volatile("" ::: "memory");
                    while (issued < avail && issued < i + 3u) ISSUE_R();
                    if (avail < i + 3u && avail < 32u) {
                        ASK();
                        THERE();
                        while (issued < avail && issued < i + 3u) ISSUE_R();
                    }
                    continue;
                }
                if (i + 4u < avail || avail == 32u) {
                    const u32 lim = avail == 32u ? 29u : avail - 5u;
                    while (i < lim) STEP();
                    if (avail < 32u) {
                        ASK();
                        STEP();
                        STEP();
                        THERE();
                    }
                    continue;
                }
                ASK();
                asm volatile("s_waitcnt vmcnt(16)\n\ts_cmp_eq_u32 %0, 3\n\ts_cbranch_scc1 .Lwd_%=\n\t"
                             "s_waitcnt vmcnt(8)\n\ts_cmp_eq_u32 %0, 2\n\ts_cbranch_scc1 .Lwd_%=\n\t"
                             "s_waitcnt vmcnt(0)\n"
                             ".Lwd_%=:" :: "s"(issued - i) : "memory", "scc");
                BARRIER();
                ++i;
                THERE();
                while (issued < avail && issued < i + 3u) ISSUE_R();
            }
            if (i <= 29u) { WAIT_VM(16); BARRIER(); }
            if (i <= 30u) { WAIT_VM(8); BARRIER(); }
            WAIT_VM(0); BARRIER();
#undef DMA4
#undef ISSUE
#undef ISSUE_R
#undef BARRIER
#undef SEEN
#undef ASK
#undef THERE
#undef STEP
        }
        __builtin_amdgcn_s_setprio(0);
    } else {

    const u32 wm = wv >> 1, wn = wv & 1u;
    const u32 g4 = (u32)lane >> 4, l15 = (u32)lane & 15u;
    u32 xa[2], wa[4][2];
#pragma unroll
    for (int ks = 0; ks < 2; ++ks) xa[ks] = (64u * wm + l15) * 128u + (((4u * ks + g4) ^ (l15 >> 1)) << 4);
#pragma unroll
    for (int ni = 0; ni < 4; ++ni)
#pragma unroll
        for (int t = 0; t < 2; ++t) {
            const u32 qq = l15 >> 2, pp = l15 & 3u, k = 8u * g4 + 4u * t + qq, ch = 8u * wn + 2u * ni + (pp >> 1);
            wa[ni][t] = LDS_W + 256u * k + ((ch ^ ((qq << 2) | ((k >> 2) & 3u))) << 4) + 8u * (pp & 1u);
        }
#pragma unroll
    for (int ni = 0; ni < 4; ++ni) asm volatile("" : "+v"(wa[ni][0]), "+v"(wa[ni][1]));
    asm volatile("" : "+v"(xa[0]), "+v"(xa[1]));
    const rsrc_t rpc = mk_rsrc(me + OFF_PK + par * PBYTES, PBYTES);
    for (; tl & 256u; tl >>= 16) {
        const u32 R = (tl >> 4) & 15u, C = tl & 15u;
        const u32 pc = (INV2[((tl & 1024u) ? 2u * (R & 1u) + wm : R >> 2) * 16u + C] & 63u) * 2u + wn;
        char* qo = p.ws[(2u * (R & 3u) + wm) & 7u];
        asm volatile("" : "+s"(qo));
        u32 base = 0u;
        f32x4 acc[4][4];
        s16x8 xf0[4], wf0[4], xf1[4], wf1[4];
#pragma unroll
        for (int mi = 0; mi < 4; ++mi) {
            xf1[mi] = (s16x8){0, 0, 0, 0, 0, 0, 0, 0};
            wf1[mi] = (s16x8){0, 0, 0, 0, 0, 0, 0, 0};
#pragma unroll
            for (int ni = 0; ni < 4; ++ni) acc[mi][ni] = (f32x4){0.f, 0.f, 0.f, 0.f};
        }
#define SB __builtin_amdgcn_sched_barrier(0)
#define RX(xf, ks, mi) xf[mi] = *(LAS s16x8*)(smem + xa[ks] + u * 16384 + (mi) * 2048)
#define RW(wf, ks, ni) do { \
            const s16x4 a_ = __builtin_amdgcn_ds_read_tr16_b64_v4i16((LAS s16x4*)(smem + wa[ni][0] + u * 16384 + (ks) * 8192)); \
            const s16x4 b_ = __builtin_amdgcn_ds_read_tr16_b64_v4i16((LAS s16x4*)(smem + wa[ni][1] + u * 16384 + (ks) * 8192)); \
            wf[ni] = __builtin_shufflevector(a_, b_, 0, 1, 2, 3, 4, 5, 6, 7); } while (0)
#define M1(xf, wf, mi, ni) acc[mi][ni] = __builtin_amdgcn_mfma_f32_16x16x32_bf16(wf[ni], xf[mi], acc[mi][ni], 0, 0, 0)
#define HALF(xc, wc, xn, wn_, ks) do { SB; \
        M1(xc, wc, 0, 0); SB; RX(xn, ks, 0); SB; M1(xc, wc, 1, 0); SB; RW(wn_, ks, 0); SB; \
        M1(xc, wc, 2, 0); SB; RX(xn, ks, 1); SB; M1(xc, wc, 3, 0); SB; RX(xn, ks, 2); SB; \
        M1(xc, wc, 0, 1); SB; RX(xn, ks, 3); SB; M1(xc, wc, 1, 1); SB; RW(wn_, ks, 1); SB; \
        M1(xc, wc, 2, 1); SB; RW(wn_, ks, 2); SB; M1(xc, wc, 3, 1); SB; RW(wn_, ks, 3); SB; \
        M1(xc, wc, 0, 2); M1(xc, wc, 1, 2); M1(xc, wc, 2, 2); M1(xc, wc, 3, 2); \
        M1(xc, wc, 0, 3); M1(xc, wc, 1, 3); M1(xc, wc, 2, 3); M1(xc, wc, 3, 3); SB; } while (0)
#pragma unroll 1
        for (u32 it = (tl & 1024u) ? 6u : 0u; it < 8u; ++it) {
#pragma unroll
            for (int u = 0; u < 4; ++u) {
                __builtin_amdgcn_s_waitcnt(0xC07F);
                __builtin_amdgcn_s_barrier();
                asm volatile("" ::: "memory");
                HALF(xf1, wf1, xf0, wf0, 0);
                HALF(xf0, wf0, xf1, wf1, 1);
            }
            if (it == 5u) {
                float mx = 0.f;
#pragma unroll
                for (int mi = 0; mi < 4; ++mi)
#pragma unroll
                    for (int ni = 0; ni < 4; ++ni) {
                        mx = __builtin_fmaxf(__builtin_fmaxf(__builtin_fabsf(acc[mi][ni][0]), __builtin_fabsf(acc[mi][ni][1])), mx);
                        mx = __builtin_fmaxf(__builtin_fmaxf(__builtin_fabsf(acc[mi][ni][2]), __builtin_fabsf(acc[mi][ni][3])), mx);
                    }
                base = window_of((__float_as_uint(mx * 1.25f) >> 23) & 255u);
            }
        }
        const u32 poison = *(volatile LAS u32*)ctl;
#pragma unroll
        for (int mi = 0; mi < 4; ++mi)
#pragma unroll
            for (int ni = 0; ni < 4; ++ni) M1(xf1, wf1, mi, ni);
#undef SB
#undef RX
#undef RW
#undef M1
#undef HALF
        if (tl & 1024u) {
            const rsrc_t r3 = mk_rsrc(me + OFF_P3, P3BYTES);
#pragma unroll
            for (int mh = 0; mh < 2; ++mh) {
#pragma unroll
                for (int f = 0; f < 4; ++f) {
                    const f32x4 a = acc[2 * mh + (f >> 1)][2 * (f & 1)], b = acc[2 * mh + (f >> 1)][2 * (f & 1) + 1];
                    __builtin_amdgcn_raw_buffer_store_b128((u32x4){cvt_pk(a[0], a[1]), cvt_pk(a[2], a[3]), cvt_pk(b[0], b[1]), cvt_pk(b[2], b[3])},
                                                           r3, ((R >> 1) * 256u + pc * 2u + mh) * 4096u + f * 1024u + lane * 16u, 0, 17);
                }
            }
            WAIT_VM(0);
            if (lane < 2) __hip_atomic_store((GAS u64*)(me + OFF_H + par * HBYTES + H2_OFF + (((rank ^ ((R >> 1) + 1u)) & 3u) * 128u + pc) * HS + 32u + lane * 8u), (u64)call, __ATOMIC_RELAXED, __HIP_MEMORY_SCOPE_SYSTEM);
            continue;
        }
        u32 hdr[2], sl[2], pl[2], d[2][16];
        High hi[2];
        const float twice = (base & H_ODD) ? 2.0f : 1.0f;
#pragma unroll
        for (int mh = 0; mh < 2; ++mh) {
#pragma unroll
            for (int mi = 0; mi < 2; ++mi)
#pragma unroll
                for (int ni = 0; ni < 4; ++ni) {
                    const f32x4 v = acc[2 * mh + mi][ni] * twice;
                    d[mh][(mi * 4 + ni) * 2] = cvt_pk(v[0], v[1]);
                    d[mh][(mi * 4 + ni) * 2 + 1] = cvt_pk(v[2], v[3]);
                }
            hdr[mh] = base;
            sl[mh] = 2048u + (2u * (R & 3u) + wm) * 256u + pc * 2u + mh;
            hi[mh] = split_high(d[mh], base);
            pl[mh] = hi[mh].a[3];
        }
        char* rec = qo + OFF_H + par * HBYTES + H2_OFF + ((rank & 3u) * 128u + pc) * HS;
        pack_planes<2>(pl, hdr, rpc, sl, smem + LDS_ENC + wv * 192u, lane, (u64)(rec + 128u + (rank >> 2) * 128u + lane * 4u));
#pragma unroll
        for (int mh = 0; mh < 2; ++mh) {
            pack_low(d[mh], rpc, sl[mh], lane);
            pack_high(hi[mh], hdr[mh], rpc, sl[mh], lane);
        }
        WAIT_VM(0);
        if (lane < 2) {
            GAS u64* g = (GAS u64*)(rec + (rank >> 2) * 16u + lane * 8u);
            __hip_atomic_store(g, ((u64)((lane ? hdr[1] : hdr[0]) | poison) << 32) | call, __ATOMIC_RELAXED, __HIP_MEMORY_SCOPE_SYSTEM);
        }
    }
    }
    if (!(tl & 512u) || 3u - (wv >> 1) >= GW) return;
    n = 512u + (3u - (wv >> 1)) * (2u * NOWN) + (tl & 31u) * 2u + (wv & 1u);
    }

    {
        const bool job = B >= NC;
        const u32 gr = wv & 1u;
        const u32 t = job ? 3u : B % 3u;
        const u32 q = (rank ^ (t + 1u)) & 3u;
        LAS u32* turn = ctl + 16u + gr * 16u;
        GAS u32* tk = (GAS u32*)(me + OFF_TK + (t * 2u + gr) * 128u);
        const u32 tbase = job ? (call - 1u) * 256u - GW * 2u * NOWN - 512u : (call - 1u) * (768u - NPP * 4u) - GW * NG - NPP * 4u;
        const rsrc_t rh = mk_rsrc(me + OFF_H + par * HBYTES, HBYTES), r3 = mk_rsrc(me + OFF_P3, P3BYTES);
        const char* pk0 = p.ws[q] + OFF_PK + par * PBYTES;
        const char* pk1 = p.ws[q | 4u] + OFF_PK + par * PBYTES;
        const char* pkg = gr ? pk1 : pk0;
        const rsrc_t ru = mk_rsrc(me + OFF_UI + par * UIBYTES, UIBYTES), ro = mk_rsrc(p.out, IN_BYTES);
        const u32 qbit = 1u << (lane >> 2), g4 = (u32)lane >> 4;
        const u32 lo = ((u32)lane & 15u) * 8192u + (((g4 & 1u) << 1) | (g4 >> 1)) * 16u;
        const char* hb = me + OFF_H + par * HBYTES;
        LAS char* dec = smem + LDS_DEC + wv * 512u;
        *(volatile LAS u64*)(dec + lane * 8) = 0ull;
        const u32 out_need = job ? 3u : 7u, y32 = lane < 32 ? lane * 4u : OOB;
        const u32 yz = lane < 32 ? 128u + g4 * 128u + gr * 56u + ((u32)lane & 15u) * 4u : OOB;
        while (n < 768u) {
            const u32 m = 0u - (n >> 9);
            auto sel = [&](u32 in, u32 out) { return in ^ ((in ^ out) & m); };
            const u32 hso = sel(((q + 4u * gr) * 64u + (n >> 3)) * H1S + (n & 6u) * 8u, H2_OFF + (q * 128u + ((n >> 1) & 127u)) * HS + gr * 8u), hd = sel(8u, 16u);
            const char* ha = hb + hso;
            u64 g0, g1, g2;
            u32 e;
#define POLL "s_load_dwordx2 %0, %4, 0x0 glc\n\ts_load_dwordx2 %1, %4, %5 glc\n\ts_load_dwordx2 %2, %4, 0x20 glc\n\t"
            asm volatile(POLL "s_load_dword %3, %6, %7\n\ts_waitcnt lgkmcnt(0)"
                         : "=&s"(g0), "=&s"(g1), "=&s"(g2), "=&s"(e) : "s"(ha), "s"(hd), "s"(&ORDS[0]), "s"(n & ~3u) : "memory");
            const u32 slot = sel((((rank >> (1u - (e >> 6))) & 1u) * 2u + (rank >> 2)) * 512u - gr, 1536u + rank * 256u) + n;
            const u32 need = uni(sel(((e >> 3) & 8u) + ((e >> 1) & 6u) + ((n >> 1) & 1u) == q + 4u * gr ? 0u : 3u, out_need));
            auto differs = [&](u64 g) { const u32 x = (u32)g ^ call; return (x | (0u - x)) >> 31; };
            u32 hs0 = (u32)(g0 >> 32), hs1 = (u32)(g1 >> 32);
            if (__builtin_expect((differs(g0) | (differs(g1) << 1) | (differs(g2) << 2)) & need, 0)) {
                const u64 t1 = __builtin_amdgcn_s_memrealtime();
                for (u32 c = 1;; ++c) {
                    u32 old;
                    __builtin_amdgcn_s_sleep(2);
                    if (c > 60u) __builtin_amdgcn_s_sleep(16);
                    if (c & 15u) {
                        u32 dummy;
                        asm volatile(POLL "s_waitcnt lgkmcnt(0)" : "=&s"(g0), "=&s"(g1), "=&s"(g2), "=&s"(dummy) : "s"(ha), "s"(hd) : "memory");
                        old = differs(g0) | (differs(g1) << 1) | (differs(g2) << 2);
                        hs0 = (u32)(g0 >> 32); hs1 = (u32)(g1 >> 32);
                    } else {
                        const u32x2 hv = __builtin_amdgcn_raw_buffer_load_b64(rh, lane < 3 ? lane * hd : OOB, hso, 17);
                        old = (u32)__builtin_amdgcn_ballot_w64(hv[0] != call);
                        hs0 = (u32)__builtin_amdgcn_readlane((int)hv[1], 0); hs1 = (u32)__builtin_amdgcn_readlane((int)hv[1], 1);
                        if (uni(__hip_atomic_load(tmo, __ATOMIC_RELAXED, __HIP_MEMORY_SCOPE_AGENT)) == call || __builtin_amdgcn_s_memrealtime() - t1 > DEADLINE) {
                            __hip_atomic_store(tmo, call, __ATOMIC_RELAXED, __HIP_MEMORY_SCOPE_AGENT);
                            old = 0u; hs0 = hs1 = H_POISON;
                        }
                    }
                    if (!(old & need)) break;
                }
            }
#undef POLL
            const u32 m0 = 0u - (need & 1u), m1 = 0u - ((need >> 1) & 1u);
            const u32 h0 = uni((hs0 & m0) | (H_DENSE & ~m0)), h1 = uni((hs1 & m1) | (H_DENSE & ~m1));
            const bool bad = ((h0 | h1) & H_POISON) != 0u;
            const u32 xo = exc_off(slot) + lane * 16u;
            u32 nb = PBYTES & ((((h0 | h1) >> 30) & 1u) - 1u), nb0 = nb & m0, nb1 = nb & m1;
            u32 so0 = slot_off(slot), so1 = so0 + (3072u & ~m), sl = list_off(slot);
            const char* pa = n < 512u ? pkg : pk0;
            const char* pb = n < 512u ? pkg : pk1;
            nb &= ~m & ((((h0 & h1) >> 24) & 1u) - 1u);
            asm volatile("" : "+s"(nb), "+s"(nb0), "+s"(nb1), "+s"(so0), "+s"(so1), "+s"(sl), "+s"(pa), "+s"(pb) :: "memory");
            u32 z1 = 0u;
            if (n >= 512u) z1 = __builtin_amdgcn_raw_buffer_load_b32(rh, yz, hso, 17);
            const rsrc_t r0 = mk_rsrc(pa, nb0), r1 = mk_rsrc(pb, nb1), rl = mk_rsrc(pa, nb);
            const u32 nanm = bad ? 0x7fc07fc0u : 0u;
            u32x4 own[4];
#pragma unroll
            for (int f = 0; f < 4; ++f) own[f] = (u32x4){nanm, nanm, nanm, nanm};
            if ((need & 4u) && !bad) {
#pragma unroll
                for (int f = 0; f < 4; ++f) own[f] = __builtin_amdgcn_raw_buffer_load_b128(r3, (t * 256u + n - 512u) * 4096u + f * 1024u + lane * 16u, 0, 17);
            }
            u32 place;
            { u32 pv_; u64 ex_;
              asm volatile("s_cmp_eq_u32 %5, 0\n\ts_cbranch_scc1 .Lfree_%=\n\ts_mov_b64 %2, exec\n\ts_mov_b64 exec, 1\n\tds_add_rtn_u32 %1, %3, %4\n\ts_waitcnt lgkmcnt(0)\n\tv_readfirstlane_b32 %0, %1\n"
                           ".Lturn_%=:\n\tds_read_b32 %1, %3 offset:4\n\ts_waitcnt lgkmcnt(0)\n\tv_cmp_ne_u32 vcc, %0, %1\n\ts_cbranch_vccnz .Lturn_%=\n\t"
                           "s_mov_b64 exec, %2\n.Lfree_%=:"
                           : "=&s"(place), "=&v"(pv_), "=&s"(ex_) : "v"((u32)(size_t)turn), "v"(1u), "s"(need) : "vcc", "scc", "memory"); }
            Blk k0, k1;
            const u32 z0 = __builtin_amdgcn_raw_buffer_load_b32(rl, y32, sl, 17);
            k0.p = __builtin_amdgcn_raw_buffer_load_b96(r0, 2048u + lane * 12u, so0, 17);
            k1.p = __builtin_amdgcn_raw_buffer_load_b96(r1, 2048u + lane * 12u, so1, 17);
            k0.l0 = __builtin_amdgcn_raw_buffer_load_b128(r0, lane * 16u, so0, 17);
            k1.l0 = __builtin_amdgcn_raw_buffer_load_b128(r1, lane * 16u, so1, 17);
            k0.l1 = __builtin_amdgcn_raw_buffer_load_b128(r0, 1024u + lane * 16u, so0, 17);
            k1.l1 = __builtin_amdgcn_raw_buffer_load_b128(r1, 1024u + lane * 16u, so1, 17);
            __builtin_amdgcn_sched_barrier(0);
            k0.b = k1.b = (u32x4){0u, 0u, 0u, 0u};
            k0.x = k1.x = 0u;
            if ((h0 | h1) & (H_DENSE | 0xffffu)) {
                k0.b = __builtin_amdgcn_raw_buffer_load_b128(r0, (h0 & qbit) ? xo : OOB, 0, 17);
                k1.b = __builtin_amdgcn_raw_buffer_load_b128(r1, (h1 & qbit) ? xo + (1024u & ~m) : OOB, 0, 17);
                k0.x = __builtin_amdgcn_raw_buffer_load_b32(r0, (h0 & H_DENSE) ? 2816u + lane * 4u : OOB, so0, 17);
                k1.x = __builtin_amdgcn_raw_buffer_load_b32(r1, (h1 & H_DENSE) ? 2816u + lane * 4u : OOB, so1, 17);
            }
            __builtin_amdgcn_sched_barrier(0);
            f32x2 ownf[16];
#pragma unroll
            for (int j = 0; j < 16; ++j) {
                ownf[j] = (f32x2){__uint_as_float(own[j >> 2][j & 3] << 16), __uint_as_float(own[j >> 2][j & 3] & 0xffff0000u)};
                asm volatile("" : "+v"(ownf[j]));
            }
            u32 e0 = 0u, e1 = 0u;
            if (n >= 512u) { e0 = plane_of(z1, 0, dec, lane); e1 = plane_of(z1, 16, dec + 256, lane); asm volatile("" : "+v"(e0), "+v"(e1)); }
            __builtin_amdgcn_sched_barrier(0);
            WAIT_VM(4);
            if (need) *(volatile LAS u32*)(turn + 1) = place + 1u;
            __builtin_amdgcn_s_waitcnt(0x0F70);
            u32 tkt, nn;
            { u64 ex_;
              asm volatile("s_mov_b64 %1, exec\n\ts_mov_b64 exec, 1\n\tglobal_atomic_add %0, %2, %3, %4 sc0\n\ts_mov_b64 exec, %1"
                           : "=&v"(tkt), "=&s"(ex_) : "v"(0u), "v"(2u), "s"(tk) : "memory"); }
            __builtin_amdgcn_sched_barrier(0);
            u32 H0[8], H1[8];
            if (n < 512u) { e0 = plane_of(z0, 0, dec, lane); e1 = plane_of(z0, 16, dec + 256, lane); }
            high_bytes(k0, (h0 & H_DENSE) ? k0.x : e0, h0, H0);
            high_bytes(k1, (h1 & H_DENSE) ? k1.x : e1, h1, H1);
            if (n < 512u) {
                const u32 uo = (e * 6u + t * 2u + gr) * 16384u + (n & 2u) * 4096u + lane * 16u;
                const u32 dn0 = ((h0 >> 23) & 1u) * 0x00800080u, dn1 = ((h1 >> 23) & 1u) * 0x00800080u;
#pragma unroll
                for (int u = 0; u < 4; ++u) __builtin_amdgcn_raw_buffer_store_b128(as_bf16(k0, H0, u) - dn0, ru, uo + u * 1024u, 0, 17);
#pragma unroll
                for (int u = 0; u < 4; ++u) __builtin_amdgcn_raw_buffer_store_b128(as_bf16(k1, H1, u) - dn1, ru, uo + 4096u + u * 1024u, 0, 17);
                asm volatile("s_waitcnt vmcnt(0)\n\tv_readfirstlane_b32 %0, %1\n\ts_nop 4" : "=s"(nn) : "v"(tkt) : "memory");
                if ((u32)lane < REP && !bad) __hip_atomic_store((GAS u32*)(me + OFF_RF + e * RFS + lane * 256u + par * 128u + ((n & 2u) * 4u + t * 2u + gr) * 4u), rtag, __ATOMIC_RELAXED, __HIP_MEMORY_SCOPE_AGENT);
            } else {
                const f32x2 sc0 = (h0 & H_ODD) ? 0.5f : 1.0f, sc1 = (h1 & H_ODD) ? 0.5f : 1.0f;
                const u32 oo = (256u * (q >> 1) + 64u * (e >> 4) + 32u * gr) * 8192u + (2048u * (q & 1u) + 128u * (e & 15u) + 32u * (n & 2u)) * 2u + lo;
#pragma unroll
                for (int f = 0; f < 4; ++f) {
                    u32 o[4];
#pragma unroll
                    for (int i = 0; i < 4; ++i) {
                        const f32x2 r = silu2(__builtin_elementwise_fma(as_f32(k1, H1, 4 * f + i), sc1, __builtin_elementwise_fma(as_f32(k0, H0, 4 * f + i), sc0, ownf[4 * f + i])));
                        o[i] = cvt_pk(r.x, r.y);
                    }
                    const auto sx = __builtin_amdgcn_permlane16_swap(o[0], o[2], false, false);
                    const auto sy = __builtin_amdgcn_permlane16_swap(o[1], o[3], false, false);
                    __builtin_amdgcn_raw_buffer_store_b128((u32x4){sx[0], sy[0], sx[1], sy[1]}, ro, oo + (f >> 1) * 131072u + (f & 1) * 64u, 0, 17);
                }
                asm volatile("s_waitcnt vmcnt(4)\n\tv_readfirstlane_b32 %0, %1\n\ts_nop 4" : "=s"(nn) : "v"(tkt) : "memory");
            }
            const bool was_in = n < 512u;
            n = nn - tbase + gr;
            if (was_in && n >= 512u) {
                ((volatile LAS u32*)turn)[2] = 1u;
                while (!uni(*(volatile LAS u32*)(ctl + 34u - gr * 16u))) __builtin_amdgcn_s_sleep(2);
            }
        }
    }
}

}

extern "C" void kernel_launch(int rank, int nranks, ncclComm_t comm, hipStream_t stream,
                              void* const* d_in, const int* in_sizes, int n_in,
                              void* d_out, int out_size, const int* in_dtypes, int out_dtype,
                              void* const* d_out_all, void* const* d_ws_all, size_t ws_bytes) {
    (void)comm; (void)in_dtypes; (void)out_dtype; (void)d_out_all;
    if (nranks != NR || (unsigned)rank >= (unsigned)NR || n_in != 2 || in_sizes[0] != 4096 * 512 || in_sizes[1] != 4096 * 512 ||
        out_size != 512 * 4096 || ws_bytes < WS_NEED) {
        fprintf(stderr, "kernel_launch: unexpected problem shape\n");
        return;
    }
    Params p;
    for (int i = 0; i < NR; ++i) p.ws[i] = (char*)d_ws_all[i];
    p.x = (const char*)d_in[0];
    p.w = (const char*)d_in[1];
    p.out = (char*)d_out;
    p.me = p.ws[rank];
    p.rank = rank;
    p.pad = 0;
    fused<<<NWG, 512, 0, stream>>>(p);
}
```
